```python
import numpy as np

kernel_name = "gpu_gemm_a2a_m4096_k4096_n16384_e5m2e5m2_dqrelu"


ABSMAX_RTOL = 0.05

COMPUTE_DTYPE = "float32"
EPILOGUE = "dequant_relu"
IN_QUANT = None
OUT_QUANT = None
IN_FP8 = "e5m2e5m2"
IN_DTYPES = {"x": "e5m2", "w_mat": "e5m2",
             "scale_x": "float32", "scale_w": "float32"}
WORLD_SIZE = 8

def _snap_fp8(x, mant_bits, max_abs, min_normal):
    a = np.abs(x.astype(np.float64))
    _m, _e = np.frexp(a)
    q = np.where(a >= min_normal, np.ldexp(1.0, _e - 1 - mant_bits),
                 min_normal * 2.0 ** -mant_bits)
    snapped = np.minimum(np.round(a / q) * q, max_abs)
    return (np.sign(x) * snapped).astype(np.float32)

def setup_inputs():

    x = _snap_fp8(np.random.randn(4096, 4096).astype(np.float32), 2, 57344.0, 2.0 ** -14)
    w_mat = _snap_fp8(np.random.randn(4096, 16384).astype(np.float32), 2, 57344.0, 2.0 ** -14)
    scale_x = (0.02 * (1.0 + np.random.rand(1))).astype(np.float32)
    scale_w = (0.02 * (1.0 + np.random.rand(1))).astype(np.float32)
    return {"x": x, "w_mat": w_mat, "scale_x": scale_x, "scale_w": scale_w}

def reference(x, w_mat, scale_x, scale_w):
    acc = (x @ w_mat).astype(np.float32)
    y = (acc * (scale_x * scale_w)).astype(np.float32)
    return np.maximum(y, 0.0).astype(np.float32)
```

```cpp
#include <hip/hip_runtime.h>
#include <rccl/rccl.h>
#include <cstdio>
#include <cstdint>

#define HIP_CHECK(expr) do { hipError_t _e = (expr); if (_e != hipSuccess) { \
    fprintf(stderr, "HIP error %s at %s:%d\n", hipGetErrorString(_e), __FILE__, __LINE__); \
    return; } } while (0)

namespace {

#define LAS __attribute__((address_space(3)))
typedef int v4i __attribute__((ext_vector_type(4)));
typedef unsigned v4u __attribute__((ext_vector_type(4)));

constexpr int NR = 8;
constexpr int KDIM = 4096, N_ALL = 16384, MSH = 512, NSH = 2048;
constexpr int BM = 128, BN = 256, BK = 128, NSTEP = KDIM / BK;
constexpr int NRBLK = MSH / BM;
constexpr int NUNIT = NRBLK * NSTEP;
constexpr int XUNIT = BM * BK, XSLOT = XUNIT + 2 * 16 * BK, WSLOT = BK * BN;
constexpr int NXS = 4;
constexpr int X_OFF = 65536, C_OFF = X_OFF + NXS * XSLOT, C_LIST = C_OFF + 1024, LDS_BYTES = C_LIST + 1536;
constexpr int C_FULL = C_OFF, C_XFREE = C_OFF + 16, C_ABORT = C_OFF + 48, C_DUMMY = C_OFF + 256;
constexpr int NPB = 32;
constexpr int NGRP = (NR - 1) * NRBLK;

constexpr size_t WS_FLAG = 0;
constexpr size_t WS_TMO = 32 << 10;
constexpr size_t WS_WCNT = 40 << 10;
constexpr size_t WS_CNT = 64 << 10;
constexpr size_t WS_X = 1 << 20;
constexpr size_t SHARD_BYTES = (size_t)MSH * KDIM;
constexpr size_t WS_W = WS_X + 2 * NR * SHARD_BYTES;
constexpr size_t WS_NEED = WS_W + (size_t)KDIM * NSH;
constexpr unsigned long long DEADLINE_TICKS = 30000000ull;
constexpr int NRAW = 32;
constexpr int HALF = XUNIT / 2, PACKED = 6 * 1024, NEXC = 128;

static_assert(LDS_BYTES <= 160 * 1024, "LDS");

constexpr int PACE_RAW = 225, PACE_PK = 178;
constexpr int WIN = 4;
constexpr int SLACK = 20;
constexpr int T_STAG = 35;
constexpr int LEADN = 4;
constexpr unsigned TR_FIRST = 100, TR_SECOND = 610, TR_PERIOD = 400;
constexpr int LAG1 = 1, LAG2 = 2, LAG3 = 3;

struct Params {
    const unsigned char* x;
    const unsigned char* w;
    const float* sx;
    const float* sw;
    float* out;
    unsigned char* ws[NR];
    unsigned char* self;
    int rank;
    int pad;
};

__device__ __forceinline__ unsigned long long rt() { return __builtin_amdgcn_s_memrealtime(); }
__device__ __forceinline__ int rfl(int v) { return __builtin_amdgcn_readfirstlane(v); }
template <int N> __device__ __forceinline__ void wait_vm() { asm volatile("s_waitcnt vmcnt(%0)" :: "n"(N) : "memory"); }

__device__ __forceinline__ int sreg(int v) {
    int r;
    asm("s_mov_b32 %0, %1" : "=s"(r) : "s"(rfl(v)));
    return r;
}
template <typename T> __device__ __forceinline__ T* uniform_ptr(T* q) {
    const unsigned long long a = (unsigned long long)q;
    unsigned long long r;
    asm("s_mov_b64 %0, %1" : "=s"(r) : "s"(((unsigned long long)(unsigned)rfl((int)(a >> 32)) << 32) | (unsigned)rfl((int)(unsigned)a)));
    return (T*)r;
}
__device__ __forceinline__ v4i make_srd(const void* p, unsigned bytes) {
    const unsigned long long a = (unsigned long long)p;
    v4i r;
    r[0] = sreg((int)(unsigned)a);
    r[1] = sreg((int)((unsigned)(a >> 32) & 0xffffu));
    r[2] = sreg((int)bytes);
    r[3] = 0x00020000;
    return r;
}
__device__ __forceinline__ unsigned ticket_issue(unsigned char* ws, size_t off, int lane) {
    unsigned v = 1u;
    asm volatile("buffer_atomic_add %0, %1, %2, 0 offen sc0" : "+v"(v) : "v"(lane * 4), "s"(make_srd(ws + off, 4)) : "memory");
    return v;
}
__device__ __forceinline__ unsigned ticket_call(unsigned v) {
    asm volatile("" : "+v"(v));
    return (unsigned)rfl((int)v) + 1u;
}
__device__ __forceinline__ unsigned bfi(unsigned m, unsigned a, unsigned b) {
    unsigned r;
    asm("v_bfi_b32 %0, %1, %2, %3" : "=v"(r) : "s"(m), "v"(a), "v"(b));
    return r;
}
__device__ __forceinline__ int wave_sum(int v) {
    v += __builtin_amdgcn_update_dpp(0, v, 0x111, 0xf, 0xf, true);
    v += __builtin_amdgcn_update_dpp(0, v, 0x112, 0xf, 0xf, true);
    v += __builtin_amdgcn_update_dpp(0, v, 0x114, 0xf, 0xf, true);
    v += __builtin_amdgcn_update_dpp(0, v, 0x118, 0xf, 0xf, true);
    return __builtin_amdgcn_readlane(v, 15) + __builtin_amdgcn_readlane(v, 31) + __builtin_amdgcn_readlane(v, 47) + __builtin_amdgcn_readlane(v, 63);
}
__device__ __forceinline__ v4i lds_quad(LAS unsigned char* L, int off) { return *(volatile LAS v4i*)(L + off); }
__device__ __forceinline__ int min4(v4i v) { return min(min(v[0], v[1]), min(v[2], v[3])); }

__device__ __forceinline__ int lag_of(int j) { return j == 0 ? 0 : j == 1 ? LAG1 : j == 2 ? LAG2 : LAG3; }
__device__ __forceinline__ int unit_pos(int j, int s) {
    const int t = s + lag_of(j);
    int p = 0;
#pragma unroll
    for (int q = 0; q < NRBLK; ++q) {
        const int d = t - lag_of(q);
        p += min(max(d, 0), NSTEP);
        p += (q < j && d < NSTEP) ? 1 : 0;
    }
    return p;
}
constexpr unsigned long long wire_head(int half) {
    const int lag[NRBLK] = {0, LAG1, LAG2, LAG3};
    unsigned long long r = 0;
    int pos = 0;
    for (int t = 0; pos < 16; ++t)
        for (int q = 0; q < NRBLK; ++q)
            if (t >= lag[q] && pos < 16) { if (pos / 8 == half) r |= (unsigned long long)(q * NSTEP + t - lag[q]) << (8 * (pos % 8)); ++pos; }
    return r;
}
constexpr unsigned long long WIRE_HEAD0 = wire_head(0), WIRE_HEAD1 = wire_head(1);
static_assert(0 <= LAG1 && LAG1 <= LAG2 && LAG2 <= LAG3, "lags must not decrease");

__device__ __forceinline__ int wire_table(LAS int* scr, int lane) {
    scr[unit_pos(lane >> 5, lane & 31)] = lane;
    scr[unit_pos(2 + (lane >> 5), lane & 31)] = 64 + lane;
    asm volatile("s_waitcnt lgkmcnt(0)" ::: "memory");
    return *(volatile LAS int*)(scr + lane) | (*(volatile LAS int*)(scr + 64 + lane) << 8);
}
__device__ __forceinline__ int unit_of(int tab, int n) {
    const int u = n >> 1;
    return (__builtin_amdgcn_readlane(tab, u & 63) >> ((u >> 6) * 8)) & (NUNIT - 1);
}
#define PIN8(q) asm volatile("" : "+v"(q[0]), "+v"(q[1]), "+v"(q[2]), "+v"(q[3]), "+v"(q[4]), "+v"(q[5]), "+v"(q[6]), "+v"(q[7]))
#define PUT(v, vo, off) asm volatile("global_store_dwordx4 %0, %1, %2 offset:%3 sc0 sc1\n\ts_nop 1" :: "v"(vo), "v"(v), "s"(dbase), "n"(off) : "memory")
__device__ __forceinline__ void load_raw(v4u (&q)[8], const unsigned char* x, int id, int h, int lane) {
    const unsigned o = (unsigned)(((id >> 5) * BM + h * 64 + (lane & 15)) * KDIM + (id & 31) * BK + (lane >> 4) * 16);
#pragma unroll
    for (int i = 0; i < 8; ++i)
        asm volatile("global_load_dwordx4 %0, %1, %2 offset:%3" : "=v"(q[i]) : "v"(o + (i >> 1) * 16 * KDIM), "s"(x), "n"((i & 1) * 64) : "memory");
}

__device__ __forceinline__ void transpose_w(const Params& p, unsigned char* ws, LAS unsigned char* scr, int cb, int grp, int lane) {
    typedef int v2i __attribute__((ext_vector_type(2)));
    const unsigned long long t0 = rt();
    const int r0 = 16 * (lane >> 4) + ((lane >> 1) & 7), f0 = ((r0 >> 1) & 3) | (((lane >> 4) & 1) << 2);
    unsigned start = TR_FIRST;
#pragma nounroll
    for (int pi = grp; pi < 4 * NSTEP; pi += NGRP) {
        while ((unsigned)(rt() - t0) < start) __builtin_amdgcn_s_sleep(4);
        start = pi == grp ? TR_SECOND : start + TR_PERIOD;
        const v4i srdw = make_srd(p.w + (size_t)((pi >> 2) * BK + (pi & 1) * 64) * N_ALL + p.rank * NSH + cb * BN + ((pi >> 1) & 1) * 128, 63 * N_ALL + 128);
#pragma unroll
        for (int i = 0; i < 8; ++i) {
            const int r = 8 * i + (lane >> 3), fr = ((r >> 1) & 3) | (((r >> 4) & 1) << 2);
            asm volatile("s_mov_b32 m0, %0\n\ts_nop 0\n\tbuffer_load_dwordx4 %1, %2, 0 offen lds"
                         :: "s"((unsigned)(uintptr_t)scr + i * 1024), "v"((unsigned)(r * N_ALL + (((lane & 7) ^ fr) * 16))), "s"(srdw) : "memory");
        }
        unsigned char* const dbase = uniform_ptr(ws + WS_W + (size_t)((cb * NSTEP + (pi >> 2)) * 4 + (pi & 2)) * 8192 + (pi & 1) * 1024);
        wait_vm<0>();
        v4u v[8];
#pragma unroll
        for (int nf = 0; nf < 8; ++nf) {
            LAS unsigned char* const a = scr + r0 * 128 + ((nf ^ f0) * 16) + 8 * (lane & 1);
            const v2i lo = __builtin_amdgcn_ds_read_tr8_b64_v2i32((LAS v2i*)a);
            const v2i hi = __builtin_amdgcn_ds_read_tr8_b64_v2i32((LAS v2i*)(a + 8 * 128));
            v[nf] = (v4u){(unsigned)lo[0], (unsigned)lo[1], (unsigned)hi[0], (unsigned)hi[1]};
        }
#pragma unroll
        for (int nf = 0; nf < 8; ++nf)
            asm volatile("global_store_dwordx4 %0, %1, %2 sc1\n\ts_nop 1" :: "v"((unsigned)(nf * 2048 + lane * 16)), "v"(v[nf]), "s"(dbase) : "memory");
        wait_vm<0>();
        if (lane == 0) __hip_atomic_fetch_add((unsigned*)(ws + WS_WCNT + (size_t)cb * 128) + (pi >> 2), 1u, __ATOMIC_RELAXED, __HIP_MEMORY_SCOPE_AGENT);
    }
}

#define MF(mt, nt, w, x) "v_mfma_f32_16x16x128_f8f6f4 v[96+16*" #mt "+4*" #nt ":96+16*" #mt "+4*" #nt "+3], v[" #w ":" #w "+7], v[" #x ":" #x "+7]" \
                         ", v[96+16*" #mt "+4*" #nt ":96+16*" #mt "+4*" #nt "+3] cbsz:1 blgp:1\n\t"
#define RXL(base, mt) "ds_read_b128 v[" #base ":" #base "+3], %[xl] offset:2048*" #mt "\n\t"
#define RXH(base, mt) "ds_read_b128 v[" #base "+4:" #base "+7], %[xl] offset:2048*" #mt "+1024\n\t"
#define RX(base, mt) RXL(base, mt) RXH(base, mt)
#define LW(r, i) ".if " #i " < 4\n\tbuffer_load_dwordx4 v[" #r ":" #r "+3], %[wv0], %[srdw], %[wo] offen offset:1024*" #i " sc1\n\t.else\n\t" \
                 "buffer_load_dwordx4 v[" #r ":" #r "+3], %[wv1], %[srdw], %[wo] offen offset:1024*(" #i "-4) sc1\n\t.endif\n\t"
#define EP_ABORT "ds_read_b32 v57, %[ct] offset:48\n\t"
#define EP_SETUP0 EP_ABORT \
    "v_mbcnt_lo_u32_b32 v60, -1, 0\n\tv_mbcnt_hi_u32_b32 v60, -1, v60\n\tv_and_b32 v61, 15, v60\n\tv_lshrrev_b32 v60, 4, v60\n\t" \
    "v_lshlrev_b32 v56, 4, v61\n\tv_lshl_add_u32 v62, v61, 8, %[stb]\n\tv_lshl_or_b32 v56, v60, 13, v56\n\t"
#define EP_ADDR(k) "v_or_b32 v63, 4*" #k ", v60\n\tv_xor_b32 v59, v63, v61\n\tv_lshl_add_u32 v63, v63, 8, %[stb]\n\t" \
    "v_lshl_add_u32 v[48+" #k "], v59, 4, v62\n\tv_lshl_add_u32 v[52+" #k "], v59, 4, v63\n\t"
#define EP_SETUP1 "v_cmp_eq_u32 vcc, 0, v57\n\tv_mov_b32 v58, 0x7fc00000\n\ts_nop 0\n\tv_cndmask_b32 v58, v58, %[sc], vcc\n\t"
#define AR(mt, nt, e) "v[96+16*" #mt "+4*" #nt "+" #e "]"
#define AR2(mt, nt, e) "v[96+16*" #mt "+4*" #nt "+" #e ":96+16*" #mt "+4*" #nt "+" #e "+1]"
#define EPMUL(mt, nt) "v_pk_mul_f32 " AR2(mt, nt, 0) ", v[58:59], " AR2(mt, nt, 0) " op_sel_hi:[0,1]\n\tv_pk_mul_f32 " AR2(mt, nt, 2) ", v[58:59], " AR2(mt, nt, 2) " op_sel_hi:[0,1]\n\t"
#define EPMAX(r) "v_maximum3_f32 " r ", " r ", 0, 0\n\t"
#define EPW(mt, nt) EPMUL(mt, nt) EPMAX(AR(mt, nt, 0)) EPMAX(AR(mt, nt, 1)) EPMAX(AR(mt, nt, 2)) EPMAX(AR(mt, nt, 3)) \
    "ds_write_b128 v[48+" #nt "], v[96+16*" #mt "+4*" #nt ":96+16*" #mt "+4*" #nt "+3] offset:(" #mt "&1)*4096\n\t"
#define EPR(mt) "ds_read_b128 v[32:35], v52 offset:(" #mt "&1)*4096\n\tds_read_b128 v[36:39], v53 offset:(" #mt "&1)*4096\n\t" \
                "ds_read_b128 v[40:43], v54 offset:(" #mt "&1)*4096\n\tds_read_b128 v[44:47], v55 offset:(" #mt "&1)*4096\n\t"
#define EPS(i) "buffer_store_dwordx4 v[32+4*" #i ":32+4*" #i "+3], v56, %[srdo], %[ob] offen sc1\n\ts_add_i32 %[ob], %[ob], 0x8000\n\t"
#define EP_OWN "s_mov_b32 %[tmp], %[ob]\n\ts_mov_b32 %[ob], %[o8]\n\ts_mov_b32 %[o8], %[tmp]\n\t"
#define EP_PEER "s_mov_b32 %[ob], %[o8]\n\t"
#define OWN_WAIT "6:\n\tds_read_b32 v12, %[ct] offset:12\n\t" LG(0) "v_readfirstlane_b32 %[tmp], v12\n\ts_cmp_ge_i32 %[tmp], %[need]\n\t" \
                 "s_cbranch_scc1 7f\n\ts_sleep 1\n\ts_branch 6b\n\t7:\n\t"
#define LG(n) "s_waitcnt lgkmcnt(" #n ")\n\t"
#define CHK "ds_read_b128 v[12:15], %[ct]\n\t" LG(0) \
            "v_min3_i32 v12, v12, v13, v14\n\tv_min_i32 v12, v12, v15\n\ts_nop 0\n\tv_readfirstlane_b32 %[tmp], v12\n\ts_cmp_ge_i32 %[tmp], %[need]\n\t"
#define ADV_ADDR \
    "s_cmp_eq_u32 %[slot], 3\n\t" \
    "s_cselect_b32 %[dlt], %[dneg], %[dpos]\n\t" \
    "s_cselect_b32 %[slot], -1, %[slot]\n\t" \
    "s_add_i32 %[slot], %[slot], 1\n\t"

#define VM(n) "s_waitcnt vmcnt(" #n ")\n\t"
#define SLOW_PATH(a, b) #a ":\n\t" CHK "s_cbranch_scc1 " #b "b\n\ts_sleep 1\n\ts_branch " #a "b\n\t"
#define CLOOP10 \
    LW(32, 0) LW(36, 1) LW(40, 2) LW(44, 3) LW(48, 4) LW(52, 5) LW(56, 6) LW(60, 7) "s_add_i32 %[wo], %[wo], 0x8000\n\t" \
    RX(16, 0) \
    "2:\n\t" \
    VM(0) \
    LG(0) MF(0, 0, 32, 16) RXL(24, 1) RXH(24, 1) \
    MF(0, 1, 40, 16) \
    MF(0, 2, 48, 16) LW(64, 0) \
    MF(0, 3, 56, 16) LW(68, 1) \
    LG(0) MF(1, 0, 32, 24) RXL(16, 2) RXH(16, 2) \
    MF(1, 1, 40, 24) \
    MF(1, 2, 48, 24) LW(72, 2) \
    MF(1, 3, 56, 24) LW(76, 3) \
    LG(0) MF(2, 0, 32, 16) RXL(24, 3) RXH(24, 3) \
    MF(2, 1, 40, 16) "s_add_i32 %[need], %[need], 1\n\t" \
    MF(2, 2, 48, 16) LW(80, 4) \
    MF(2, 3, 56, 16) LW(84, 5) \
    LG(0) MF(3, 0, 32, 24) RXL(16, 4) RXH(16, 4) \
    MF(3, 1, 40, 24) ADV_ADDR \
    MF(3, 2, 48, 24) LW(88, 6) \
    MF(3, 3, 56, 24) LW(92, 7) \
    LG(0) MF(4, 0, 32, 16) RXL(24, 5) RXH(24, 5) \
    MF(4, 1, 40, 16) \
    MF(4, 2, 48, 16) \
    MF(4, 3, 56, 16) \
    LG(0) MF(5, 0, 32, 24) RXL(16, 6) RXH(16, 6) \
    MF(5, 1, 40, 24) \
    MF(5, 2, 48, 24) \
    MF(5, 3, 56, 24) \
    LG(0) MF(6, 0, 32, 16) RXL(24, 7) RXH(24, 7) \
    MF(6, 1, 40, 16) \
    MF(6, 2, 48, 16) \
    MF(6, 3, 56, 16) \
    LG(0) MF(7, 0, 32, 24) RXL(16, 8) RXH(16, 8) \
    MF(7, 1, 40, 24) \
    MF(7, 2, 48, 24) "ds_read_b128 v[12:15], %[ct]\n\t" \
    MF(7, 3, 56, 24) \
    LG(1) MF(8, 0, 32, 16) RXL(24, 9) RXH(24, 9) \
    MF(8, 1, 40, 16) \
    MF(8, 2, 48, 16) "ds_write_b32 %[xf], %[cnt]\n\t" LG(3) "v_min3_i32 v12, v12, v13, v14\n\tv_min_i32 v12, v12, v15\n\t" \
    MF(8, 3, 56, 16) "v_add_u32 %[xl], %[dlt], %[xl]\n\t" "v_readfirstlane_b32 %[tmp], v12\n\t" "v_add_u32 %[cnt], 1, %[cnt]\n\t" \
    "s_cmp_eq_u32 %[need], 32\n\ts_cbranch_scc1 5f\n\t" \
    "s_cmp_ge_i32 %[tmp], %[need]\n\ts_cbranch_scc0 14f\n\t" \
    "13:\n\t" \
    LG(1) MF(9, 0, 32, 24) RXL(16, 0) RXH(16, 0) \
    MF(9, 1, 40, 24) \
    MF(9, 2, 48, 24) \
    MF(9, 3, 56, 24) "s_add_i32 %[wo], %[wo], 0x8000\n\t" \
    VM(0) \
    LG(0) MF(0, 0, 64, 16) RXL(24, 1) RXH(24, 1) \
    MF(0, 1, 72, 16) \
    MF(0, 2, 80, 16) LW(32, 0) \
    MF(0, 3, 88, 16) LW(36, 1) \
    LG(0) MF(1, 0, 64, 24) RXL(16, 2) RXH(16, 2) \
    MF(1, 1, 72, 24) \
    MF(1, 2, 80, 24) LW(40, 2) \
    MF(1, 3, 88, 24) LW(44, 3) \
    LG(0) MF(2, 0, 64, 16) RXL(24, 3) RXH(24, 3) \
    MF(2, 1, 72, 16) "s_add_i32 %[need], %[need], 1\n\t" \
    MF(2, 2, 80, 16) LW(48, 4) \
    MF(2, 3, 88, 16) LW(52, 5) \
    LG(0) MF(3, 0, 64, 24) RXL(16, 4) RXH(16, 4) \
    MF(3, 1, 72, 24) ADV_ADDR \
    MF(3, 2, 80, 24) LW(56, 6) \
    MF(3, 3, 88, 24) LW(60, 7) \
    LG(0) MF(4, 0, 64, 16) RXL(24, 5) RXH(24, 5) \
    MF(4, 1, 72, 16) \
    MF(4, 2, 80, 16) \
    MF(4, 3, 88, 16) \
    LG(0) MF(5, 0, 64, 24) RXL(16, 6) RXH(16, 6) \
    MF(5, 1, 72, 24) \
    MF(5, 2, 80, 24) \
    MF(5, 3, 88, 24) \
    LG(0) MF(6, 0, 64, 16) RXL(24, 7) RXH(24, 7) \
    MF(6, 1, 72, 16) \
    MF(6, 2, 80, 16) \
    MF(6, 3, 88, 16) \
    LG(0) MF(7, 0, 64, 24) RXL(16, 8) RXH(16, 8) \
    MF(7, 1, 72, 24) \
    MF(7, 2, 80, 24) "ds_read_b128 v[12:15], %[ct]\n\t" \
    MF(7, 3, 88, 24) \
    LG(1) MF(8, 0, 64, 16) RXL(24, 9) RXH(24, 9) \
    MF(8, 1, 72, 16) \
    MF(8, 2, 80, 16) "ds_write_b32 %[xf], %[cnt]\n\t" LG(3) "v_min3_i32 v12, v12, v13, v14\n\tv_min_i32 v12, v12, v15\n\t" \
    MF(8, 3, 88, 16) "v_add_u32 %[xl], %[dlt], %[xl]\n\t" "v_readfirstlane_b32 %[tmp], v12\n\t" "v_add_u32 %[cnt], 1, %[cnt]\n\t" \
    "s_cmp_ge_i32 %[tmp], %[need]\n\ts_cbranch_scc0 4f\n\t" \
    "3:\n\t" \
    LG(1) MF(9, 0, 64, 24) RXL(16, 0) RXH(16, 0) \
    MF(9, 1, 72, 24) \
    MF(9, 2, 80, 24) \
    MF(9, 3, 88, 24) "s_add_i32 %[wo], %[wo], 0x8000\n\t" \
    "s_branch 2b\n\t" \
    SLOW_PATH(14, 13) \
    SLOW_PATH(4, 3) \
    "5:\n\t" LG(0) \
    MF(9, 0, 32, 24) MF(9, 1, 40, 24) MF(9, 2, 48, 24) MF(9, 3, 56, 24) \
    OWN_WAIT VM(0) \
    RX(16, 8) RX(24, 9) EP_SETUP0 EP_ADDR(0) EP_ADDR(1) EP_ADDR(2) EP_ADDR(3) LG(0) EP_SETUP1 \
    MF(8, 0, 64, 16) MF(8, 1, 72, 16) MF(8, 2, 80, 16) MF(8, 3, 88, 16) \
    MF(9, 0, 64, 24) MF(9, 1, 72, 24) MF(9, 2, 80, 24) MF(9, 3, 88, 24) \
    EP_OWN "s_nop 7\n\ts_nop 7\n\t" \
    EPW(8, 0) EPW(8, 1) EPW(8, 2) EPW(8, 3) EPR(8) LG(0) EPS(0) EPS(1) EPS(2) EPS(3) \
    EPW(9, 0) EPW(9, 1) EPW(9, 2) EPW(9, 3) EPR(9) LG(0) EPS(0) EPS(1) EPS(2) EPS(3) \
    "8:\n\t" CHK "s_cbranch_scc1 9f\n\ts_sleep 1\n\ts_branch 8b\n\t9:\n\t" \
    RX(16, 0) EP_ABORT EP_PEER \
    LG(1) MF(0, 0, 64, 16) RXL(24, 1) RXH(24, 1) \
    MF(0, 1, 72, 16) \
    MF(0, 2, 80, 16) LG(2) EP_SETUP1 \
    MF(0, 3, 88, 16) "s_nop 3\n\t" \
    LG(0) MF(1, 0, 64, 24) RXL(16, 2) RXH(16, 2) EPW(0, 0) \
    MF(1, 1, 72, 24) EPW(0, 1) \
    MF(1, 2, 80, 24) EPW(0, 2) \
    MF(1, 3, 88, 24) EPW(0, 3) EPR(0) \
    LG(8) MF(2, 0, 64, 16) RXL(24, 3) RXH(24, 3) EPW(1, 0) \
    MF(2, 1, 72, 16) EPW(1, 1) \
    MF(2, 2, 80, 16) EPW(1, 2) LG(5) EPS(0) EPS(1) \
    MF(2, 3, 88, 16) EPW(1, 3) EPS(2) EPS(3) EPR(1) \
    LG(8) MF(3, 0, 64, 24) RXL(16, 4) RXH(16, 4) EPW(2, 0) \
    MF(3, 1, 72, 24) EPW(2, 1) \
    MF(3, 2, 80, 24) EPW(2, 2) LG(5) EPS(0) EPS(1) \
    MF(3, 3, 88, 24) EPW(2, 3) EPS(2) EPS(3) EPR(2) \
    LG(8) MF(4, 0, 64, 16) RXL(24, 5) RXH(24, 5) EPW(3, 0) \
    MF(4, 1, 72, 16) EPW(3, 1) \
    MF(4, 2, 80, 16) EPW(3, 2) LG(5) EPS(0) EPS(1) \
    MF(4, 3, 88, 16) EPW(3, 3) EPS(2) EPS(3) EPR(3) \
    LG(8) MF(5, 0, 64, 24) RXL(16, 6) RXH(16, 6) EPW(4, 0) \
    MF(5, 1, 72, 24) EPW(4, 1) \
    MF(5, 2, 80, 24) EPW(4, 2) LG(5) EPS(0) EPS(1) \
    MF(5, 3, 88, 24) EPW(4, 3) EPS(2) EPS(3) EPR(4) \
    LG(8) MF(6, 0, 64, 16) RXL(24, 7) RXH(24, 7) EPW(5, 0) \
    MF(6, 1, 72, 16) EPW(5, 1) \
    MF(6, 2, 80, 16) EPW(5, 2) LG(5) EPS(0) EPS(1) \
    MF(6, 3, 88, 16) EPW(5, 3) EPS(2) EPS(3) EPR(5) \
    LG(8) MF(7, 0, 64, 24) EPW(6, 0) \
    MF(7, 1, 72, 24) EPW(6, 1) \
    MF(7, 2, 80, 24) EPW(6, 2) LG(3) EPS(0) EPS(1) \
    MF(7, 3, 88, 24) EPW(6, 3) EPS(2) EPS(3) EPR(6) \
    "s_nop 7\n\t" EPW(7, 0) EPW(7, 1) EPW(7, 2) \
    LG(3) EPS(0) EPS(1) EPS(2) EPS(3) EPW(7, 3) EPR(7) \
    LG(0) EPS(0) EPS(1) EPS(2) EPS(3) \
    "s_branch 90f\n\t"
#define CLOOP9 \
    LW(32, 0) LW(36, 1) LW(40, 2) LW(44, 3) LW(48, 4) LW(52, 5) LW(56, 6) LW(60, 7) "s_add_i32 %[wo], %[wo], 0x8000\n\t" \
    RX(16, 0) RX(24, 1) \
    "2:\n\t" \
    VM(0) \
    LG(2) MF(0, 0, 32, 16) RXL(240, 2) \
    MF(0, 1, 40, 16) RXH(240, 2) \
    MF(0, 2, 48, 16) LW(64, 0) \
    MF(0, 3, 56, 16) LW(68, 1) \
    LG(2) MF(1, 0, 32, 24) RXL(16, 3) \
    MF(1, 1, 40, 24) RXH(16, 3) \
    MF(1, 2, 48, 24) LW(72, 2) \
    MF(1, 3, 56, 24) LW(76, 3) \
    LG(2) MF(2, 0, 32, 240) RXL(24, 4) \
    MF(2, 1, 40, 240) RXH(24, 4) "s_add_i32 %[need], %[need], 1\n\t" \
    MF(2, 2, 48, 240) LW(80, 4) \
    MF(2, 3, 56, 240) LW(84, 5) \
    LG(2) MF(3, 0, 32, 16) RXL(240, 5) \
    MF(3, 1, 40, 16) RXH(240, 5) ADV_ADDR \
    MF(3, 2, 48, 16) LW(88, 6) \
    MF(3, 3, 56, 16) LW(92, 7) \
    LG(2) MF(4, 0, 32, 24) RXL(16, 6) \
    MF(4, 1, 40, 24) RXH(16, 6) \
    MF(4, 2, 48, 24) \
    MF(4, 3, 56, 24) \
    LG(2) MF(5, 0, 32, 240) RXL(24, 7) \
    MF(5, 1, 40, 240) RXH(24, 7) \
    MF(5, 2, 48, 240) "ds_read_b128 v[12:15], %[ct]\n\t" \
    MF(5, 3, 56, 240) \
    LG(3) MF(6, 0, 32, 16) RXL(240, 8) \
    MF(6, 1, 40, 16) RXH(240, 8) \
    MF(6, 2, 48, 16) "ds_write_b32 %[xf], %[cnt]\n\t" LG(3) "v_min3_i32 v12, v12, v13, v14\n\tv_min_i32 v12, v12, v15\n\t" \
    MF(6, 3, 56, 16) "v_add_u32 %[xl], %[dlt], %[xl]\n\t" "v_readfirstlane_b32 %[tmp], v12\n\t" "v_add_u32 %[cnt], 1, %[cnt]\n\t" \
    "s_cmp_eq_u32 %[need], 32\n\ts_cbranch_scc1 5f\n\t" \
    "s_cmp_ge_i32 %[tmp], %[need]\n\ts_cbranch_scc0 14f\n\t" \
    "13:\n\t" \
    MF(7, 0, 32, 24) RXL(16, 0) \
    MF(7, 1, 40, 24) RXH(16, 0) \
    MF(7, 2, 48, 24) \
    MF(7, 3, 56, 24) \
    LG(3) MF(8, 0, 32, 240) RXL(24, 1) \
    MF(8, 1, 40, 240) RXH(24, 1) \
    MF(8, 2, 48, 240) \
    MF(8, 3, 56, 240) "s_add_i32 %[wo], %[wo], 0x8000\n\t" \
    VM(0) \
    LG(2) MF(0, 0, 64, 16) RXL(240, 2) \
    MF(0, 1, 72, 16) RXH(240, 2) \
    MF(0, 2, 80, 16) LW(32, 0) \
    MF(0, 3, 88, 16) LW(36, 1) \
    LG(2) MF(1, 0, 64, 24) RXL(16, 3) \
    MF(1, 1, 72, 24) RXH(16, 3) \
    MF(1, 2, 80, 24) LW(40, 2) \
    MF(1, 3, 88, 24) LW(44, 3) \
    LG(2) MF(2, 0, 64, 240) RXL(24, 4) \
    MF(2, 1, 72, 240) RXH(24, 4) "s_add_i32 %[need], %[need], 1\n\t" \
    MF(2, 2, 80, 240) LW(48, 4) \
    MF(2, 3, 88, 240) LW(52, 5) \
    LG(2) MF(3, 0, 64, 16) RXL(240, 5) \
    MF(3, 1, 72, 16) RXH(240, 5) ADV_ADDR \
    MF(3, 2, 80, 16) LW(56, 6) \
    MF(3, 3, 88, 16) LW(60, 7) \
    LG(2) MF(4, 0, 64, 24) RXL(16, 6) \
    MF(4, 1, 72, 24) RXH(16, 6) \
    MF(4, 2, 80, 24) \
    MF(4, 3, 88, 24) \
    LG(2) MF(5, 0, 64, 240) RXL(24, 7) \
    MF(5, 1, 72, 240) RXH(24, 7) \
    MF(5, 2, 80, 240) "ds_read_b128 v[12:15], %[ct]\n\t" \
    MF(5, 3, 88, 240) \
    LG(3) MF(6, 0, 64, 16) RXL(240, 8) \
    MF(6, 1, 72, 16) RXH(240, 8) \
    MF(6, 2, 80, 16) "ds_write_b32 %[xf], %[cnt]\n\t" LG(3) "v_min3_i32 v12, v12, v13, v14\n\tv_min_i32 v12, v12, v15\n\t" \
    MF(6, 3, 88, 16) "v_add_u32 %[xl], %[dlt], %[xl]\n\t" "v_readfirstlane_b32 %[tmp], v12\n\t" "v_add_u32 %[cnt], 1, %[cnt]\n\t" \
    "s_cmp_ge_i32 %[tmp], %[need]\n\ts_cbranch_scc0 4f\n\t" \
    "3:\n\t" \
    MF(7, 0, 64, 24) RXL(16, 0) \
    MF(7, 1, 72, 24) RXH(16, 0) \
    MF(7, 2, 80, 24) \
    MF(7, 3, 88, 24) \
    LG(3) MF(8, 0, 64, 240) RXL(24, 1) \
    MF(8, 1, 72, 240) RXH(24, 1) \
    MF(8, 2, 80, 240) \
    MF(8, 3, 88, 240) "s_add_i32 %[wo], %[wo], 0x8000\n\t" \
    "s_branch 2b\n\t" \
    SLOW_PATH(14, 13) \
    SLOW_PATH(4, 3) \
    "5:\n\t" LG(0) \
    MF(7, 0, 32, 24) MF(7, 1, 40, 24) MF(7, 2, 48, 24) MF(7, 3, 56, 24) \
    MF(8, 0, 32, 240) MF(8, 1, 40, 240) MF(8, 2, 48, 240) MF(8, 3, 56, 240) \
    OWN_WAIT VM(0) \
    RX(16, 8) EP_SETUP0 EP_ADDR(0) EP_ADDR(1) EP_ADDR(2) EP_ADDR(3) LG(0) EP_SETUP1 \
    MF(8, 0, 64, 16) MF(8, 1, 72, 16) MF(8, 2, 80, 16) MF(8, 3, 88, 16) \
    EP_OWN "s_nop 7\n\ts_nop 7\n\t" \
    EPW(8, 0) EPW(8, 1) EPW(8, 2) EPW(8, 3) EPR(8) LG(0) EPS(0) EPS(1) EPS(2) EPS(3) \
    "8:\n\t" CHK "s_cbranch_scc1 9f\n\ts_sleep 1\n\ts_branch 8b\n\t9:\n\t" \
    RX(16, 0) RX(24, 1) EP_ABORT EP_PEER \
    LG(3) MF(0, 0, 64, 16) RXL(240, 2) \
    MF(0, 1, 72, 16) RXH(240, 2) \
    MF(0, 2, 80, 16) LG(2) EP_SETUP1 \
    MF(0, 3, 88, 16) "s_nop 3\n\t" \
    MF(1, 0, 64, 24) RXL(16, 3) EPW(0, 0) \
    MF(1, 1, 72, 24) RXH(16, 3) EPW(0, 1) \
    MF(1, 2, 80, 24) EPW(0, 2) \
    MF(1, 3, 88, 24) EPW(0, 3) EPR(0) \
    LG(10) MF(2, 0, 64, 240) RXL(24, 4) EPW(1, 0) \
    MF(2, 1, 72, 240) RXH(24, 4) EPW(1, 1) \
    MF(2, 2, 80, 240) EPW(1, 2) LG(5) EPS(0) EPS(1) \
    MF(2, 3, 88, 240) EPW(1, 3) EPS(2) EPS(3) EPR(1) \
    MF(3, 0, 64, 16) RXL(240, 5) EPW(2, 0) \
    MF(3, 1, 72, 16) RXH(240, 5) EPW(2, 1) \
    MF(3, 2, 80, 16) EPW(2, 2) LG(5) EPS(0) EPS(1) \
    MF(3, 3, 88, 16) EPW(2, 3) EPS(2) EPS(3) EPR(2) \
    MF(4, 0, 64, 24) RXL(16, 6) EPW(3, 0) \
    MF(4, 1, 72, 24) RXH(16, 6) EPW(3, 1) \
    MF(4, 2, 80, 24) EPW(3, 2) LG(5) EPS(0) EPS(1) \
    MF(4, 3, 88, 24) EPW(3, 3) EPS(2) EPS(3) EPR(3) \
    MF(5, 0, 64, 240) RXL(24, 7) EPW(4, 0) \
    MF(5, 1, 72, 240) RXH(24, 7) EPW(4, 1) \
    MF(5, 2, 80, 240) EPW(4, 2) LG(5) EPS(0) EPS(1) \
    MF(5, 3, 88, 240) EPW(4, 3) EPS(2) EPS(3) EPR(4) \
    MF(6, 0, 64, 16) EPW(5, 0) \
    MF(6, 1, 72, 16) EPW(5, 1) \
    MF(6, 2, 80, 16) EPW(5, 2) LG(3) EPS(0) EPS(1) \
    MF(6, 3, 88, 16) EPW(5, 3) EPS(2) EPS(3) EPR(5) \
    MF(7, 0, 64, 24) EPW(6, 0) \
    MF(7, 1, 72, 24) EPW(6, 1) \
    MF(7, 2, 80, 24) EPW(6, 2) LG(3) EPS(0) EPS(1) \
    MF(7, 3, 88, 24) EPW(6, 3) EPS(2) EPS(3) EPR(6) \
    "s_nop 7\n\t" EPW(7, 0) EPW(7, 1) EPW(7, 2) \
    LG(3) EPS(0) EPS(1) EPS(2) EPS(3) EPW(7, 3) EPR(7) \
    LG(0) EPS(0) EPS(1) EPS(2) EPS(3) \
    "s_branch 90f\n\t"

#define V12_63 \
    "v12", "v13", "v14", "v15", "v16", "v17", "v18", "v19", "v20", "v21", "v22", "v23", "v24", "v25", "v26", "v27", "v28", \
    "v29", "v30", "v31", "v32", "v33", "v34", "v35", "v36", "v37", "v38", "v39", "v40", "v41", "v42", "v43", "v44", "v45", \
    "v46", "v47", "v48", "v49", "v50", "v51", "v52", "v53", "v54", "v55", "v56", "v57", "v58", "v59", "v60", "v61", "v62", \
    "v63"
#define V64_207 \
    "v64", "v65", "v66", "v67", "v68", "v69", "v70", "v71", "v72", "v73", "v74", "v75", "v76", "v77", "v78", "v79", "v80", \
    "v81", "v82", "v83", "v84", "v85", "v86", "v87", "v88", "v89", "v90", "v91", "v92", "v93", "v94", "v95", "v96", "v97", \
    "v98", "v99", "v100", "v101", "v102", "v103", "v104", "v105", "v106", "v107", "v108", "v109", "v110", "v111", "v112", \
    "v113", "v114", "v115", "v116", "v117", "v118", "v119", "v120", "v121", "v122", "v123", "v124", "v125", "v126", "v127", \
    "v128", "v129", "v130", "v131", "v132", "v133", "v134", "v135", "v136", "v137", "v138", "v139", "v140", "v141", "v142", \
    "v143", "v144", "v145", "v146", "v147", "v148", "v149", "v150", "v151", "v152", "v153", "v154", "v155", "v156", "v157", \
    "v158", "v159", "v160", "v161", "v162", "v163", "v164", "v165", "v166", "v167", "v168", "v169", "v170", "v171", "v172", \
    "v173", "v174", "v175", "v176", "v177", "v178", "v179", "v180", "v181", "v182", "v183", "v184", "v185", "v186", "v187", \
    "v188", "v189", "v190", "v191", "v192", "v193", "v194", "v195", "v196", "v197", "v198", "v199", "v200", "v201", "v202", \
    "v203", "v204", "v205", "v206", "v207"
#define V208_255 \
    "v208", "v209", "v210", "v211", "v212", "v213", "v214", "v215", "v216", "v217", "v218", "v219", "v220", "v221", "v222", \
    "v223", "v224", "v225", "v226", "v227", "v228", "v229", "v230", "v231", "v232", "v233", "v234", "v235", "v236", "v237", \
    "v238", "v239", "v240", "v241", "v242", "v243", "v244", "v245", "v246", "v247", "v248", "v249", "v250", "v251", "v252", \
    "v253", "v254", "v255"

__global__ void __launch_bounds__(512, 2) fused_gemm(Params p) {
    extern __shared__ __attribute__((aligned(16))) unsigned char lds_raw[];
    LAS unsigned char* L = (LAS unsigned char*)lds_raw;
    const unsigned lds0 = (unsigned)(uintptr_t)L;
    const int tid = threadIdx.x, lane = tid & 63, wave = rfl(tid >> 6);
    const int b = blockIdx.x;
    unsigned char* ws = p.self;
    asm volatile("" :: "s"(p.x), "s"(ws), "s"(p.rank));

    if (__builtin_expect(b < NPB, 1)) {
        if (b >= 4 * (NR - 1)) return;
        if (tid < 4) ((LAS int*)(L + 8 * 512))[tid] = 0;
        asm volatile("s_waitcnt lgkmcnt(0)" ::: "memory");
        __builtin_amdgcn_s_barrier();
        const unsigned long long tstart = rt();
        const int n0 = b / (NR - 1) + 4 * wave, h = n0 & 1;
        const unsigned char* const xb = uniform_ptr(p.x);
        v4u q[8];
        while ((unsigned)(rt() - tstart) < (unsigned)(wave * T_STAG)) __builtin_amdgcn_s_sleep(1);
        load_raw(q, xb, (int)((n0 < 16 ? WIRE_HEAD0 : WIRE_HEAD1) >> (4 * (n0 & 14))) & (NUNIT - 1), h, lane);
        LAS int* const ctl = (LAS int*)(L + 8 * 512);
        const unsigned t = ticket_issue(ws, WS_CNT + (size_t)(512 + b * 8 + wave) * 128, lane);
        const int tab = wire_table((LAS int*)(L + wave * 512), lane);
        const int dst = (p.rank + 1 + b % (NR - 1)) & (NR - 1);
        wait_vm<0>();
        PIN8(q);
        const unsigned call = ticket_call(t);
        const size_t xo = WS_X + (size_t)((call & 1) * NR + p.rank) * SHARD_BYTES + h * HALF;
        const size_t fo = WS_FLAG + (size_t)((call & 1) * NR + p.rank) * 1024 + h * 128;
        unsigned char* const dbase = uniform_ptr(p.ws[dst] + xo);
        unsigned char* const sbase = uniform_ptr(ws + xo);
        const unsigned tb = (unsigned)rfl((int)(unsigned)tstart);
        const int ui = (n0 >> 1) + 16 * (lane & 7), idl = (__builtin_amdgcn_ds_bpermute((ui & 63) * 4, tab) >> ((ui >> 6) * 8)) & (NUNIT - 1);
        const unsigned* const flp = (const unsigned*)(ws + fo) + (idl >> 5) * 64 + (idl & 31);
        unsigned fl = 0;
#pragma nounroll
        for (int n = n0; n < 2 * NUNIT; n += 32) {
            const int id = unit_of(tab, n);
            const unsigned o = (unsigned)id * XUNIT + lane * 16;
            unsigned hdr = 0x80u;
            const int m = n >> 2;
            if (n != n0) {
                hdr = (unsigned)__builtin_amdgcn_readlane((int)fl, n >> 5);
                if ((hdr >> 8) != call) {
                    while (rfl(*(volatile LAS int*)ctl) < m - LEADN) __builtin_amdgcn_s_sleep(2);
                    fl = __hip_atomic_load(flp, __ATOMIC_RELAXED, __HIP_MEMORY_SCOPE_SYSTEM);
                    hdr = (unsigned)__builtin_amdgcn_readlane((int)fl, n >> 5);
                }
                hdr = (hdr >> 8) == call ? hdr & 0xffu : 0x80u;
                if (hdr & 0x80u) {
                    load_raw(q, xb, id, h, lane);
                    wait_vm<0>();
                    PIN8(q);
                } else {
                    asm volatile("global_load_dwordx4 %0, %7, %9 sc0 sc1\n\t"
                                 "global_load_dwordx4 %1, %7, %9 offset:1024 sc0 sc1\n\t"
                                 "global_load_dwordx4 %2, %7, %9 offset:2048 sc0 sc1\n\t"
                                 "global_load_dwordx4 %3, %7, %9 offset:3072 sc0 sc1\n\t"
                                 "global_load_dwordx4 %4, %8, %9 sc0 sc1\n\t"
                                 "global_load_dwordx4 %5, %8, %9 offset:1024 sc0 sc1\n\t"
                                 "global_load_dwordx4 %6, %8, %9 offset:2048 sc0 sc1\n\t"
                                 "s_waitcnt vmcnt(0)"
                                 : "=&v"(q[0]), "=&v"(q[1]), "=&v"(q[2]), "=&v"(q[3]), "=&v"(q[4]), "=&v"(q[5]), "=&v"(q[6])
                                 : "v"(o), "v"(o + 4096), "s"(sbase) : "memory");
                }
            }
            unsigned now;
            v4i cv;
            for (;;) {
                cv = *(volatile LAS v4i*)ctl;
                now = ((unsigned)rt() - tb) * 16u;
                if (rfl(cv[0]) == m && rfl(cv[1]) >= m - (WIN - 1) && (int)(now - (unsigned)rfl(cv[2])) >= 0) break;
                __builtin_amdgcn_s_sleep(1);
            }
            PUT(q[0], o, 0); PUT(q[1], o, 1024); PUT(q[2], o, 2048); PUT(q[3], o, 3072);
            PUT(q[4], o + 4096, 0); PUT(q[5], o + 4096, 1024);
            if (hdr & 0x80u) { PUT(q[6], o + 4096, 2048); PUT(q[7], o + 4096, 3072); }
            else if (lane < 16) PUT(q[6], o + 4096, 2048);
            if (lane == 0) {
                *(volatile LAS int*)(ctl + 2) = max(cv[2], (int)now - SLACK * 16) + ((hdr & 0x80u) ? 4 * PACE_RAW : 4 * PACE_PK);
                *(volatile LAS int*)ctl = m + 1;
            }
            wait_vm<0>();
            if (lane == 0) {
                __hip_atomic_store((unsigned*)(p.ws[dst] + fo) + (id >> 5) * 64 + (id & 31), (call << 8) | hdr, __ATOMIC_RELAXED, __HIP_MEMORY_SCOPE_SYSTEM);
                __hip_atomic_fetch_add(ctl + 1, 1, __ATOMIC_RELAXED, __HIP_MEMORY_SCOPE_WORKGROUP);
            }
        }
        return;
    }

    const int bi = b - NPB;
    const int cb = 4 * (b & 1) + ((bi >> 3) & 3), grp = ((b >> 1) & 3) * 7 + (bi >> 5);
    const int j = grp & (NRBLK - 1);
    const int src = (p.rank + 1 + (grp >> 2)) & (NR - 1);
    const int nloc = j == 0 && grp < 16 ? 2 : 1;
    const int f8 = grp + min((grp + 3) >> 2, 4);

    if (tid < 64) *(LAS int*)(L + C_OFF + 4 * tid) = 0;
    asm volatile("s_waitcnt lgkmcnt(0)" ::: "memory");
    __builtin_amdgcn_s_barrier();
    asm volatile("" ::: "memory");

    if (wave < 4) {
        if (wave == 1) {
            static_assert(NRAW + 256 - NPB == 2 * NUNIT, "one half unit per GEMM block");
            const int n = NRAW + bi, h = n & 1;
            LAS unsigned short* const list = (LAS unsigned short*)(L + C_LIST);
            const unsigned t = ticket_issue(ws, WS_CNT + (size_t)(768 + bi) * 128, lane);
            const int id = unit_of(wire_table((LAS int*)list, lane), n);
            v4u q[8];
            load_raw(q, uniform_ptr(p.x), id, h, lane);
            wait_vm<0>();
            PIN8(q);
            const unsigned call = ticket_call(t);
            unsigned sum = 0;
#pragma unroll
            for (int c = 0; c < 4; ++c) sum = __builtin_amdgcn_sad_u8(q[c][c] & 0x7f7f7f7fu, 0u, sum);
            const unsigned B = (unsigned)rfl(min(max((wave_sum((int)sum) - 18944) >> 12, 0), 24));
            const unsigned b4 = B * 0x04040404u;
            *(LAS unsigned*)(list + 2 * lane) = (unsigned)(lane << 6) | (unsigned)((2 * lane + 1) << 21);
            int cnt = 0;
            v4u tt[8], pk[6];
            unsigned m[8];
#pragma unroll
            for (int i = 0; i < 8; ++i) {
                m[i] = 0;
#pragma unroll
                for (int c = 0; c < 4; ++c) {
                    const unsigned x = q[i][c];
                    tt[i][c] = (x | 0x80808080u) - b4;
                    const unsigned dl = (((tt[i][c] & 0x1f1f1f1fu) + b4) ^ x) & 0x60606060u;
                    m[i] |= c == 3 ? dl << 1 : dl >> (5 - 2 * c);
                }
            }
#pragma unroll
            for (int g2 = 0; g2 < 2; ++g2)
#pragma unroll
                for (int c = 0; c < 4; ++c) {
                    const unsigned D = bfi(0x80808080u, q[4 * g2 + 3][c], tt[4 * g2 + 3][c]);
                    pk[3 * g2 + 0][c] = bfi(0x60606060u, D << 2, bfi(0x80808080u, q[4 * g2 + 0][c], tt[4 * g2 + 0][c]));
                    pk[3 * g2 + 1][c] = bfi(0x60606060u, D << 4, bfi(0x80808080u, q[4 * g2 + 1][c], tt[4 * g2 + 1][c]));
                    pk[3 * g2 + 2][c] = bfi(0x60606060u, __builtin_amdgcn_alignbit(D, D, 2), bfi(0x80808080u, q[4 * g2 + 2][c], tt[4 * g2 + 2][c]));
                }
            unsigned char* const dbase = uniform_ptr(ws + WS_X + (size_t)((call & 1) * NR + p.rank) * SHARD_BYTES + h * HALF);
            const unsigned o = (unsigned)id * XUNIT + lane * 16;
            PUT(pk[0], o, 0); PUT(pk[1], o, 1024); PUT(pk[2], o, 2048); PUT(pk[3], o, 3072);
            PUT(pk[4], o + 4096, 0); PUT(pk[5], o + 4096, 1024);
            const unsigned list0 = (unsigned)(uintptr_t)list, dummy = list0 + 2 * (NEXC + 512) + 2 * lane;
            do {
                unsigned long long act[8];
                unsigned base[8];
#pragma unroll
                for (int i = 0; i < 8; ++i) act[i] = __ballot(m[i] != 0);
#pragma unroll
                for (int i = 0; i < 8; ++i) { base[i] = list0 + 2 * cnt; cnt += __popcll(act[i]); }
#pragma unroll
                for (int i = 0; i < 8; ++i) {
                    unsigned bt, ad;
                    asm("v_ffbl_b32 %0, %1" : "=v"(bt) : "v"(m[i]));
                    bt &= 30u;
                    const unsigned d = __builtin_amdgcn_ubfe(m[i], bt, 2u);
                    m[i] ^= d << bt;
                    const unsigned rk = __builtin_amdgcn_mbcnt_hi((unsigned)(act[i] >> 32), __builtin_amdgcn_mbcnt_lo((unsigned)act[i], 0u));
                    asm("v_cndmask_b32 %0, %1, %2, %3" : "=v"(ad) : "v"(dummy), "v"(base[i] + 2 * rk), "s"(act[i]));
                    *(LAS unsigned short*)(uintptr_t)ad = (unsigned short)((d << 14) | (unsigned)((i * 64 + lane) << 5) | bt);
                }
            } while (cnt <= NEXC && __ballot((m[0] | m[1] | m[2] | m[3] | m[4] | m[5] | m[6] | m[7]) != 0) != 0);
            const v4u ex = *(LAS v4u*)(list + 8 * (lane & 15));
            if (lane < 16) PUT(ex, o + 4096, 2048);
            wait_vm<0>();
            if (lane == 0)
                __hip_atomic_store((unsigned*)(ws + WS_FLAG + (size_t)((call & 1) * NR + p.rank) * 1024 + h * 128) + (id >> 5) * 64 + (id & 31),
                                   (call << 8) | (cnt <= NEXC ? B : 0x80u), __ATOMIC_RELAXED, __HIP_MEMORY_SCOPE_SYSTEM);
        }
        float sc = p.sx[0] * p.sw[0];
        asm volatile("" : "+v"(sc));
        unsigned vx = lds0 + X_OFF + lane * 16;
        const unsigned vxf = lds0 + (lane == 0 ? C_XFREE + 4 * wave : C_DUMMY + 4 * lane);
        const unsigned vct = lds0 + C_FULL;
        const v4i srdw = make_srd(ws + WS_W + (size_t)cb * NSTEP * WSLOT + wave * 8192, NSTEP * WSLOT - wave * 8192);
        const v4i srdo = make_srd(p.out, (unsigned)(NR * MSH) * NSH * 4u);
        const int ocol = (cb * BN + wave * 64) * 4;
        int ob = sreg(ocol + (src * MSH + j * BM) * NSH * 4);
        int o8 = sreg(ocol + (p.rank * MSH + f8 * 16) * NSH * 4);
        int cnt = 1, need = 1, slot = 0, wo = 0, dlt, tmp;
        static_assert(C_ABORT - C_FULL == 48 && 4 * NSH * 4 == 0x8000 && WSLOT == 0x8000, "literals in the consumer");
        asm volatile(
            "s_setprio 2\n\t"
            ".set i_, 96\n\t.rept 160\n\tv_mov_b32 v[i_], 0\n\t.set i_, i_+1\n\t.endr\n\t"
            "1:\n\t"
            "s_sleep 1\n\t"
            CHK "s_cbranch_scc0 1b\n\t"
            "s_cmp_eq_u32 %[nloc], 1\n\ts_cbranch_scc1 81f\n\t"
            CLOOP10
            "81:\n\t"
            CLOOP9
            "90:\n\t"
            : [xl] "+&v"(vx), [cnt] "+&v"(cnt),
              [need] "+&s"(need), [slot] "+&s"(slot), [wo] "+&s"(wo), [ob] "+&s"(ob), [o8] "+&s"(o8), [dlt] "=&s"(dlt), [tmp] "=&s"(tmp)
            : [xf] "v"(vxf), [ct] "v"(vct), [wv0] "v"(lane * 16), [wv1] "v"(lane * 16 + 4096), [sc] "v"(sc), [srdw] "s"(srdw), [srdo] "s"(srdo),
              [stb] "s"(lds0 + wave * 8192), [nloc] "s"(rfl(nloc)),
              [dpos] "s"(XSLOT), [dneg] "s"(-(NXS - 1) * XSLOT)
            : "memory", "scc", "vcc",
              V12_63, V64_207, V208_255);
    } else if (wave == 4 || wave == 7) {
        const int h = wave == 7;
        const unsigned t = ticket_issue(ws, WS_CNT + (size_t)(b * 2 + h) * 128, lane);
        wait_vm<0>();
        const unsigned call = ticket_call(t);
        const v4i srd = make_srd(ws + WS_X + ((size_t)((call & 1) * NR + src) * NUNIT + j * NSTEP) * XUNIT + h * HALF, NSTEP * XUNIT - h * HALF);
        unsigned char* const wsb = uniform_ptr(ws);
        const unsigned vfl = lane < 32 ? (unsigned)(WS_FLAG + (((call & 1) * NR + src) * NRBLK + j) * 256 + h * 128 + lane * 4) : (unsigned)WS_TMO;
        const unsigned vd0 = lane * 16, vd1 = lane * 16 + 4096, vex = lane * 4 + PACKED;
        const unsigned vfull = lds0 + (lane == 0 ? C_FULL + 4 * h : C_DUMMY + 4 * lane);
        const unsigned vab = lds0 + (lane == 0 ? C_ABORT : C_DUMMY + 4 * lane);
        const unsigned vxf = lds0 + C_XFREE;
        static_assert(NXS == 4, "the decoder is unrolled over the x ring");
        const int sl0 = (int)(lds0 + X_OFF + h * HALF), sl2 = sl0 + 2 * XSLOT;
        const unsigned vs0 = sl0 + lane * 16, vs2 = sl2 + lane * 16;
        int ss = 0, nn = 0, fr = 0, t0, t1, t2;
        asm volatile(
            ".macro DEC_ISSUE k\n\t"
            "s_lshl_b32 %[t0], %[n], 14\n\t"
            "buffer_load_dwordx4 v[64+28*\\k:64+28*\\k+3], %[vd0], %[srd], %[t0] offen sc0 sc1\n\t"
            "buffer_load_dwordx4 v[64+28*\\k+4:64+28*\\k+7], %[vd0], %[srd], %[t0] offen offset:1024 sc0 sc1\n\t"
            "buffer_load_dwordx4 v[64+28*\\k+8:64+28*\\k+11], %[vd0], %[srd], %[t0] offen offset:2048 sc0 sc1\n\t"
            "buffer_load_dwordx4 v[64+28*\\k+12:64+28*\\k+15], %[vd0], %[srd], %[t0] offen offset:3072 sc0 sc1\n\t"
            "buffer_load_dwordx4 v[64+28*\\k+16:64+28*\\k+19], %[vd1], %[srd], %[t0] offen sc0 sc1\n\t"
            "buffer_load_dwordx4 v[64+28*\\k+20:64+28*\\k+23], %[vd1], %[srd], %[t0] offen offset:1024 sc0 sc1\n\t"
            "buffer_load_dword v[64+28*\\k+24], %[vex], %[srd], %[t0] offen sc0 sc1\n\t"
            "global_load_dword v[64+28*\\k+25], %[vfl], %[wsb] sc0 sc1\n\t"
            "s_add_i32 %[n], %[n], 1\n\t"
            ".endm\n\t"
            ".macro DEC_RDY\n\t"
            "v_lshrrev_b32 v189, 7, v188\n\t"
            "v_cmp_eq_u32 vcc, %[call2], v189\n\t"
            "s_mov_b32 s66, vcc_lo\n\t"
            ".endm\n\t"
            ".macro DEC_STEP k, kn, va, sa\n\t"
            "s_cmp_gt_i32 %[n], %[s]\n\t"
            "s_cbranch_scc0 .Ldec_a\\k\n\t"
            ".Ldec_b\\k:\n\t"
            "s_add_i32 %[t2], %[s], -3\n\t"
            "s_cmp_ge_i32 %[fr], %[t2]\n\t"
            "s_cbranch_scc0 .Ldec_free\\k\n\t"
            ".Ldec_c\\k:\n\t"
            "v_readlane_b32 %[t1], v188, %[s]\n\t"
            "ds_read_b128 v[208:211], %[vxf]\n\t"
            "s_add_i32 %[s], %[s], 1\n\t"
            "s_and_b32 %[t1], %[t1], 31\n\t"
            "s_mul_i32 %[t1], %[t1], 0x04040404\n\t"
            "s_lshl_b32 %[t2], %[s], 14\n\t"
            "s_waitcnt vmcnt(0)\n\t"
            "v_max_u32 v188, v188, v[64+28*\\k+25]\n\t"
            "v_lshrrev_b32 v185, 16, v[64+28*\\k+24]\n\t"
            "DEC_RDY\n\t"
            "v_and_b32 v[192], %[m60], v[64+28*\\k+0]\n\t"
            "v_and_b32 v[193], %[m60], v[64+28*\\k+1]\n\t"
            "v_and_b32 v[194], %[m60], v[64+28*\\k+2]\n\t"
            "v_and_b32 v[195], %[m60], v[64+28*\\k+3]\n\t"
            "s_bitcmp1_b64 s[66:67], %[s]\n\t"
            "s_cselect_b32 %[t0], %[t2], %[far]\n\t"
            "s_addc_u32 %[n], %[n], 0\n\t"
            "v_and_b32 v[196], %[m60], v[64+28*\\k+4]\n\t"
            "v_and_b32 v[197], %[m60], v[64+28*\\k+5]\n\t"
            "v_and_b32 v[198], %[m60], v[64+28*\\k+6]\n\t"
            "v_and_b32 v[199], %[m60], v[64+28*\\k+7]\n\t"
            "buffer_load_dwordx4 v[64+28*\\kn:64+28*\\kn+3], %[vd0], %[srd], %[t0] offen sc0 sc1\n\t"
            "v_and_b32 v[200], %[m60], v[64+28*\\k+8]\n\t"
            "v_and_b32 v[201], %[m60], v[64+28*\\k+9]\n\t"
            "v_and_b32 v[202], %[m60], v[64+28*\\k+10]\n\t"
            "v_and_b32 v[203], %[m60], v[64+28*\\k+11]\n\t"
            "v_xad_u32 v[64+28*\\k+0], v[64+28*\\k+0], v[192], %[t1]\n\t"
            "buffer_load_dwordx4 v[64+28*\\kn+4:64+28*\\kn+7], %[vd0], %[srd], %[t0] offen offset:1024 sc0 sc1\n\t"
            "v_xad_u32 v[64+28*\\k+1], v[64+28*\\k+1], v[193], %[t1]\n\t"
            "v_xad_u32 v[64+28*\\k+2], v[64+28*\\k+2], v[194], %[t1]\n\t"
            "v_xad_u32 v[64+28*\\k+3], v[64+28*\\k+3], v[195], %[t1]\n\t"
            "ds_write_b128 \\va, v[64+28*\\k:64+28*\\k+3] offset:(\\k&1)*20480\n\t"
            "v_lshrrev_b32 v[192], 2, v[192]\n\t"
            "v_lshrrev_b32 v[193], 2, v[193]\n\t"
            "buffer_load_dwordx4 v[64+28*\\kn+8:64+28*\\kn+11], %[vd0], %[srd], %[t0] offen offset:2048 sc0 sc1\n\t"
            "v_lshrrev_b32 v[194], 2, v[194]\n\t"
            "v_lshrrev_b32 v[195], 2, v[195]\n\t"
            "v_xad_u32 v[64+28*\\k+4], v[64+28*\\k+4], v[196], %[t1]\n\t"
            "v_xad_u32 v[64+28*\\k+5], v[64+28*\\k+5], v[197], %[t1]\n\t"
            "v_xad_u32 v[64+28*\\k+6], v[64+28*\\k+6], v[198], %[t1]\n\t"
            "buffer_load_dwordx4 v[64+28*\\kn+12:64+28*\\kn+15], %[vd0], %[srd], %[t0] offen offset:3072 sc0 sc1\n\t"
            "v_xad_u32 v[64+28*\\k+7], v[64+28*\\k+7], v[199], %[t1]\n\t"
            "ds_write_b128 \\va, v[64+28*\\k+4:64+28*\\k+7] offset:(\\k&1)*20480+1024\n\t"
            "v_lshrrev_b32 v[196], 4, v[196]\n\t"
            "v_lshrrev_b32 v[197], 4, v[197]\n\t"
            "v_lshrrev_b32 v[198], 4, v[198]\n\t"
            "v_lshrrev_b32 v[199], 4, v[199]\n\t"
            "buffer_load_dwordx4 v[64+28*\\kn+16:64+28*\\kn+19], %[vd1], %[srd], %[t0] offen sc0 sc1\n\t"
            "v_xad_u32 v[64+28*\\k+8], v[64+28*\\k+8], v[200], %[t1]\n\t"
            "v_xad_u32 v[64+28*\\k+9], v[64+28*\\k+9], v[201], %[t1]\n\t"
            "v_xad_u32 v[64+28*\\k+10], v[64+28*\\k+10], v[202], %[t1]\n\t"
            "v_xad_u32 v[64+28*\\k+11], v[64+28*\\k+11], v[203], %[t1]\n\t"
            "ds_write_b128 \\va, v[64+28*\\k+8:64+28*\\k+11] offset:(\\k&1)*20480+2048\n\t"
            "buffer_load_dwordx4 v[64+28*\\kn+20:64+28*\\kn+23], %[vd1], %[srd], %[t0] offen offset:1024 sc0 sc1\n\t"
            "v_alignbit_b32 v[200], v[200], v[200], 30\n\t"
            "v_alignbit_b32 v[201], v[201], v[201], 30\n\t"
            "v_alignbit_b32 v[202], v[202], v[202], 30\n\t"
            "v_alignbit_b32 v[203], v[203], v[203], 30\n\t"
            "v_add3_u32 v[192], v[192], v[196], v[200]\n\t"
            "buffer_load_dword v[64+28*\\kn+24], %[vex], %[srd], %[t0] offen sc0 sc1\n\t"
            "v_add3_u32 v[193], v[193], v[197], v[201]\n\t"
            "v_add3_u32 v[194], v[194], v[198], v[202]\n\t"
            "v_add3_u32 v[195], v[195], v[199], v[203]\n\t"
            "v_add_u32 v[176], %[t1], v[192]\n\t"
            "v_add_u32 v[177], %[t1], v[193]\n\t"
            "global_load_dword v[64+28*\\kn+25], %[vfl], %[wsb] sc0 sc1\n\t"
            "v_add_u32 v[178], %[t1], v[194]\n\t"
            "v_add_u32 v[179], %[t1], v[195]\n\t"
            "ds_write_b128 \\va, v[176:179] offset:(\\k&1)*20480+3072\n\t"
            "v_bfe_u32 v180, v[64+28*\\k+24], 5, 9\n\t"
            "v_bfe_u32 v181, v185, 5, 9\n\t"
            "v_and_b32 v182, 6, v[64+28*\\k+24]\n\t"
            "v_and_b32 v183, 6, v185\n\t"
            "v_lshl_add_u32 v180, v180, 4, \\sa\n\t"
            "v_lshl_add_u32 v181, v181, 4, \\sa\n\t"
            "v_lshl_add_u32 v180, v182, 1, v180\n\t"
            "v_lshl_add_u32 v181, v183, 1, v181\n\t"
            "v_and_or_b32 v182, v[64+28*\\k+24], 24, 5\n\t"
            "v_and_or_b32 v183, v185, 24, 5\n\t"
            "v_bfe_u32 v186, v[64+28*\\k+24], 14, 2\n\t"
            "v_lshrrev_b32 v185, 14, v185\n\t"
            "v_lshlrev_b32 v186, v182, v186\n\t"
            "v_lshlrev_b32 v185, v183, v185\n\t"
            "v_mov_b32 v187, %[s]\n\t"
            "s_waitcnt lgkmcnt(4)\n\t"
            "v_min3_i32 v208, v208, v209, v210\n\t"
            "v_min_i32 v208, v208, v211\n\t"
            "v_and_b32 v[192], %[m60], v[64+28*\\k+12+0]\n\t"
            "v_readfirstlane_b32 %[fr], v208\n\t"
            "v_and_b32 v[193], %[m60], v[64+28*\\k+12+1]\n\t"
            "v_and_b32 v[194], %[m60], v[64+28*\\k+12+2]\n\t"
            "v_and_b32 v[195], %[m60], v[64+28*\\k+12+3]\n\t"
            "v_and_b32 v[196], %[m60], v[64+28*\\k+12+4]\n\t"
            "v_and_b32 v[197], %[m60], v[64+28*\\k+12+5]\n\t"
            "v_and_b32 v[198], %[m60], v[64+28*\\k+12+6]\n\t"
            "v_and_b32 v[199], %[m60], v[64+28*\\k+12+7]\n\t"
            "v_and_b32 v[200], %[m60], v[64+28*\\k+12+8]\n\t"
            "v_and_b32 v[201], %[m60], v[64+28*\\k+12+9]\n\t"
            "v_and_b32 v[202], %[m60], v[64+28*\\k+12+10]\n\t"
            "v_and_b32 v[203], %[m60], v[64+28*\\k+12+11]\n\t"
            "v_xad_u32 v[64+28*\\k+12+0], v[64+28*\\k+12+0], v[192], %[t1]\n\t"
            "v_xad_u32 v[64+28*\\k+12+1], v[64+28*\\k+12+1], v[193], %[t1]\n\t"
            "v_xad_u32 v[64+28*\\k+12+2], v[64+28*\\k+12+2], v[194], %[t1]\n\t"
            "v_xad_u32 v[64+28*\\k+12+3], v[64+28*\\k+12+3], v[195], %[t1]\n\t"
            "ds_write_b128 \\va, v[64+28*\\k+12:64+28*\\k+12+3] offset:(\\k&1)*20480+4096\n\t"
            "v_lshrrev_b32 v[192], 2, v[192]\n\t"
            "v_lshrrev_b32 v[193], 2, v[193]\n\t"
            "v_lshrrev_b32 v[194], 2, v[194]\n\t"
            "v_lshrrev_b32 v[195], 2, v[195]\n\t"
            "v_xad_u32 v[64+28*\\k+12+4], v[64+28*\\k+12+4], v[196], %[t1]\n\t"
            "v_xad_u32 v[64+28*\\k+12+5], v[64+28*\\k+12+5], v[197], %[t1]\n\t"
            "v_xad_u32 v[64+28*\\k+12+6], v[64+28*\\k+12+6], v[198], %[t1]\n\t"
            "v_xad_u32 v[64+28*\\k+12+7], v[64+28*\\k+12+7], v[199], %[t1]\n\t"
            "ds_write_b128 \\va, v[64+28*\\k+12+4:64+28*\\k+12+7] offset:(\\k&1)*20480+4096+1024\n\t"
            "v_lshrrev_b32 v[196], 4, v[196]\n\t"
            "v_lshrrev_b32 v[197], 4, v[197]\n\t"
            "v_lshrrev_b32 v[198], 4, v[198]\n\t"
            "v_lshrrev_b32 v[199], 4, v[199]\n\t"
            "v_xad_u32 v[64+28*\\k+12+8], v[64+28*\\k+12+8], v[200], %[t1]\n\t"
            "v_xad_u32 v[64+28*\\k+12+9], v[64+28*\\k+12+9], v[201], %[t1]\n\t"
            "v_xad_u32 v[64+28*\\k+12+10], v[64+28*\\k+12+10], v[202], %[t1]\n\t"
            "v_xad_u32 v[64+28*\\k+12+11], v[64+28*\\k+12+11], v[203], %[t1]\n\t"
            "ds_write_b128 \\va, v[64+28*\\k+12+8:64+28*\\k+12+11] offset:(\\k&1)*20480+4096+2048\n\t"
            "v_alignbit_b32 v[200], v[200], v[200], 30\n\t"
            "v_alignbit_b32 v[201], v[201], v[201], 30\n\t"
            "v_alignbit_b32 v[202], v[202], v[202], 30\n\t"
            "v_alignbit_b32 v[203], v[203], v[203], 30\n\t"
            "v_add3_u32 v[192], v[192], v[196], v[200]\n\t"
            "v_add3_u32 v[193], v[193], v[197], v[201]\n\t"
            "v_add3_u32 v[194], v[194], v[198], v[202]\n\t"
            "v_add3_u32 v[195], v[195], v[199], v[203]\n\t"
            "v_add_u32 v[204], %[t1], v[192]\n\t"
            "v_add_u32 v[205], %[t1], v[193]\n\t"
            "v_add_u32 v[206], %[t1], v[194]\n\t"
            "v_add_u32 v[207], %[t1], v[195]\n\t"
            "ds_write_b128 \\va, v[204:207] offset:(\\k&1)*20480+4096+3072\n\t"
            "ds_xor_b32 v180, v186 offset:(\\k&1)*20480\n\t"
            "ds_xor_b32 v181, v185 offset:(\\k&1)*20480\n\t"
            ".Ldec_e\\k:\n\t"
            ".if \\k == 3\n\t"
            "s_cmp_eq_u32 %[s], 32\n\t"
            "s_cbranch_scc1 45f\n\t"
            ".endif\n\t"
            "ds_write_b32 %[vfull], v187\n\t"
            ".endm\n\t"
            ".macro DEC_COLD k, sa\n\t"
            ".Ldec_a\\k:\n\t"
            "s_waitcnt vmcnt(0)\n\t"
            "v_max_u32 v188, v188, v[64+28*\\k+25]\n\t"
            "s_call_b64 s[58:59], 21b\n\t"
            "s_cbranch_scc0 1f\n\t"
            "DEC_ISSUE \\k\n\t"
            "s_branch .Ldec_b\\k\n\t"
            "1:\n\t"
            "s_call_b64 s[56:57], 24b\n\t"
            "s_lshl_b32 %[t0], %[s], 14\n\t"
            "s_add_i32 m0, \\sa, (\\k&1)*20480\n\t"
            "s_add_i32 %[n], %[n], 1\n\t"
            "buffer_load_dwordx4 %[vd0], %[srd], %[t0] offen sc0 sc1 lds\n\t"
            "buffer_load_dwordx4 %[vd0], %[srd], %[t0] offen offset:1024 sc0 sc1 lds\n\t"
            "buffer_load_dwordx4 %[vd0], %[srd], %[t0] offen offset:2048 sc0 sc1 lds\n\t"
            "buffer_load_dwordx4 %[vd0], %[srd], %[t0] offen offset:3072 sc0 sc1 lds\n\t"
            "s_add_i32 m0, \\sa, (\\k&1)*20480+4096\n\t"
            "s_add_i32 %[s], %[s], 1\n\t"
            "buffer_load_dwordx4 %[vd1], %[srd], %[t0] offen sc0 sc1 lds\n\t"
            "buffer_load_dwordx4 %[vd1], %[srd], %[t0] offen offset:1024 sc0 sc1 lds\n\t"
            "buffer_load_dwordx4 %[vd1], %[srd], %[t0] offen offset:2048 sc0 sc1 lds\n\t"
            "buffer_load_dwordx4 %[vd1], %[srd], %[t0] offen offset:3072 sc0 sc1 lds\n\t"
            "global_load_dword v188, %[vfl], %[wsb] sc0 sc1\n\t"
            "v_mov_b32 v187, %[s]\n\t"
            "s_waitcnt vmcnt(0)\n\t"
            "s_branch .Ldec_e\\k\n\t"
            ".Ldec_free\\k:\n\t"
            "s_call_b64 s[56:57], 24b\n\t"
            "s_branch .Ldec_c\\k\n\t"
            ".endm\n\t"
            "s_setprio 3\n\t"
            "s_mov_b32 s67, 0\n\t"
            "v_mov_b32 v188, 0\n\ts_mov_b32 s66, 0\n\t"
            "v_mov_b32 v[64+25], 0\n\tv_mov_b32 v[64+28+25], 0\n\tv_mov_b32 v[64+56+25], 0\n\tv_mov_b32 v[64+84+25], 0\n\t"
            "s_branch 40f\n\t"
            "21:\n\t"
            "s_nop 0\n\t"
            "v_readlane_b32 %[t2], v188, %[s]\n\t"
            "s_lshr_b32 %[t1], %[t2], 8\n\t"
            "s_cmp_eq_u32 %[t1], %[call]\n\t"
            "s_cbranch_scc1 23f\n\t"
            "s_memrealtime s[62:63]\n\t"
            "22:\n\t"
            "global_load_dword v188, %[vfl], %[wsb] sc0 sc1\n\t"
            "s_waitcnt vmcnt(0)\n\t"
            "v_readlane_b32 %[t2], v188, %[s]\n\t"
            "s_lshr_b32 %[t1], %[t2], 8\n\t"
            "s_cmp_eq_u32 %[t1], %[call]\n\t"
            "s_cbranch_scc1 27f\n\t"
            "v_readlane_b32 %[t1], v188, 32\n\t"
            "s_cmp_eq_u32 %[t1], %[call]\n\t"
            "s_cbranch_scc1 50f\n\t"
            "s_memrealtime s[60:61]\n\t"
            "s_waitcnt lgkmcnt(0)\n\t"
            "s_sub_u32 %[t1], s60, s62\n\t"
            "s_cmp_gt_u32 %[t1], %[dead]\n\t"
            "s_cbranch_scc1 50f\n\t"
            "s_sleep 1\n\t"
            "s_branch 22b\n\t"
            "27:\n\t"
            "s_waitcnt lgkmcnt(0)\n\t"
            "23:\n\t"
            "s_bitcmp0_b32 %[t2], 7\n\t"
            "s_setpc_b64 s[58:59]\n\t"
            "24:\n\t"
            "s_add_i32 %[t2], %[s], -3\n\t"
            "25:\n\t"
            "ds_read_b128 v[180:183], %[vxf]\n\t"
            "s_waitcnt lgkmcnt(0)\n\t"
            "v_min3_i32 v180, v180, v181, v182\n\t"
            "v_min_i32 v180, v180, v183\n\t"
            "s_nop 0\n\t"
            "v_readfirstlane_b32 %[fr], v180\n\t"
            "s_cmp_ge_i32 %[fr], %[t2]\n\t"
            "s_cbranch_scc0 26f\n\t"
            "s_setpc_b64 s[56:57]\n\t"
            "26:\n\t"
            "s_sleep 1\n\t"
            "s_branch 25b\n\t"
            "DEC_COLD 0, %[sl0]\n\t"
            "DEC_COLD 1, %[sl0]\n\t"
            "DEC_COLD 2, %[sl2]\n\t"
            "DEC_COLD 3, %[sl2]\n\t"
            "40:\n\t"
            "DEC_STEP 0, 1, %[vs0], %[sl0]\n\t"
            "DEC_STEP 1, 2, %[vs0], %[sl0]\n\t"
            "DEC_STEP 2, 3, %[vs2], %[sl2]\n\t"
            "DEC_STEP 3, 0, %[vs2], %[sl2]\n\t"
            "s_branch 40b\n\t"
            "45:\n\t"
            "v_readlane_b32 %[t1], v188, 32\n\t"
            "s_cmp_eq_u32 %[t1], %[call]\n\t"
            "s_cbranch_scc0 60f\n\t"
            "50:\n\t"
            "v_mov_b32 v184, %[call]\n\t"
            "v_mov_b32 v189, %[tmo]\n\t"
            "global_store_dword v189, v184, %[wsb] sc1\n\t"
            "ds_write_b32 %[vab], v184\n\t"
            "60:\n\t"
            "v_mov_b32 v187, 32\n\t"
            "ds_write_b32 %[vfull], v187\n\t"
            ".purgem DEC_ISSUE\n\t.purgem DEC_RDY\n\t.purgem DEC_STEP\n\t.purgem DEC_COLD\n\t"
            : [s] "+&s"(ss), [n] "+&s"(nn), [fr] "+&s"(fr), [t0] "=&s"(t0), [t1] "=&s"(t1), [t2] "=&s"(t2)
            : [vd0] "v"(vd0), [vd1] "v"(vd1), [vex] "v"(vex), [vfl] "v"(vfl), [vfull] "v"(vfull), [vab] "v"(vab), [vxf] "v"(vxf),
              [vs0] "v"(vs0), [vs2] "v"(vs2), [far] "s"(0x40000000), [sl0] "s"(sl0), [sl2] "s"(sl2), [m60] "s"(0x60606060),
              [srd] "s"(srd), [wsb] "s"(wsb), [call] "s"(call), [call2] "s"(call << 1), [dead] "s"((unsigned)DEADLINE_TICKS),
              [tmo] "n"((int)WS_TMO)
            : "memory", "scc", "vcc", "s56", "s57", "s58", "s59", "s60", "s61", "s62", "s63", "s66", "s67",
              V64_207, "v208", "v209", "v210", "v211");
        static_assert(XSLOT == 20480, "literal in the decoder");

    } else if (wave == 5) {
        if (lane == 0) *(volatile LAS int*)(L + C_FULL + 8) = NSTEP;
        transpose_w(p, ws, L + 4 * 8192, cb, grp, lane);
    } else {
        const unsigned t = ticket_issue(ws, WS_CNT + (size_t)(1024 + b) * 128, lane);
        LAS int* const full = (LAS int*)(L + (lane == 0 ? C_FULL + 12 : C_DUMMY + 4 * lane));
        const v4i srdx = make_srd(p.x + (size_t)f8 * 16 * KDIM, nloc * 16 * KDIM);
        const v4i srdw = make_srd(ws + WS_W + (size_t)cb * NSTEP * WSLOT, NSTEP * WSLOT);
        const unsigned vox = (unsigned)((lane & 15) * KDIM + (lane >> 4) * 16);
        const unsigned vow = (unsigned)((grp & 3) * 8192 + lane * 128);
        unsigned* const tmo = (unsigned*)(ws + WS_TMO);
        const unsigned* const pp = lane < 32 ? (const unsigned*)(ws + WS_WCNT + (size_t)cb * 128) + lane : tmo;
        wait_vm<0>();
        const unsigned call = ticket_call(t);
        unsigned junk = 0;
        int wr = 0, tc = 0;
        auto publish = [&](int landed) { *(volatile LAS int*)full = wr == NSTEP ? landed : min(landed, wr - 1); };
        auto gate = [&](int need, int landed) {
            const unsigned long long t0 = rt();
            while (wr < need) {
                const unsigned v = __hip_atomic_load(pp, __ATOMIC_RELAXED, __HIP_MEMORY_SCOPE_AGENT);
                wr = __builtin_ctzll(~(unsigned long long)(unsigned)__ballot(v >= call * 4u));
                if (wr >= need) break;
                publish(landed);
                if ((__ballot(v == call) >> 32) != 0 || rt() - t0 > DEADLINE_TICKS) {
                    __hip_atomic_store(tmo, call, __ATOMIC_RELAXED, __HIP_MEMORY_SCOPE_AGENT);
                    if (lane == 0) *(volatile LAS int*)(L + C_ABORT) = 1;
                    wr = NSTEP;
                    break;
                }
                __builtin_amdgcn_s_sleep(2);
            }
            asm volatile("" ::: "memory");
        };
        for (int s = 0; s < NSTEP; ++s) {
            if (s >= NXS && rfl(min4(lds_quad(L, C_XFREE))) < s - (NXS - 1)) {
                wait_vm<0>();
                publish(s);
                while (rfl(min4(lds_quad(L, C_XFREE))) < s - (NXS - 1)) __builtin_amdgcn_s_sleep(1);
            }
            asm volatile("" ::: "memory");
            const unsigned la = lds0 + X_OFF + (s & (NXS - 1)) * XSLOT + XUNIT;
#pragma unroll
            for (int i = 0; i < 4; ++i)
                asm volatile("s_mov_b32 m0, %0\n\ts_nop 0\n\tbuffer_load_dwordx4 %1, %2, %3 offen lds"
                             :: "s"(la + i * 1024), "v"(vox), "s"(srdx), "s"(s * BK + (i >> 1) * 16 * KDIM + (i & 1) * 64) : "memory");
            if (wr < min(s + 2, NSTEP)) {
                gate(min(s + 2, NSTEP), s + 1);
                publish(s + 1);
            } else {
                wait_vm<4>();
                publish(s);
            }
            for (; tc < min(wr, s + 2); ++tc)
                asm volatile("buffer_load_dword %0, %1, %2, %3 offen sc1" : "+v"(junk) : "v"(vow), "s"(srdw), "s"(tc * WSLOT) : "memory");
        }
        wait_vm<0>();
        publish(NSTEP);
        asm volatile("" :: "v"(junk));
    }
}

}

extern "C" void kernel_launch(int rank, int nranks, ncclComm_t comm, hipStream_t stream,
                              void* const* d_in, const int* in_sizes, int n_in,
                              void* d_out, int out_size, const int* in_dtypes, int out_dtype,
                              void* const* d_out_all, void* const* d_ws_all, size_t ws_bytes) {
    (void)comm; (void)in_dtypes; (void)out_dtype; (void)d_out_all;
    if (nranks != NR || (unsigned)rank >= (unsigned)NR || n_in != 4 || ws_bytes < WS_NEED ||
        in_sizes[0] != MSH * KDIM || in_sizes[1] != KDIM * N_ALL || out_size != NR * MSH * NSH) {
        fprintf(stderr, "kernel_launch: unexpected problem shape\n");
        return;
    }
    static bool ready[NR] = {};
    if (!ready[rank]) {
        HIP_CHECK(hipFuncSetAttribute((const void*)fused_gemm, hipFuncAttributeMaxDynamicSharedMemorySize, LDS_BYTES));
        ready[rank] = true;
    }
    Params p;
    p.x = (const unsigned char*)d_in[0];
    p.w = (const unsigned char*)d_in[1];
    p.sx = (const float*)d_in[2];
    p.sw = (const float*)d_in[3];
    p.out = (float*)d_out;
    for (int i = 0; i < NR; ++i) p.ws[i] = (unsigned char*)d_ws_all[i];
    p.self = p.ws[rank];
    p.rank = rank;
    p.pad = 0;
    fused_gemm<<<dim3(256), dim3(512), LDS_BYTES, stream>>>(p);
    HIP_CHECK(hipGetLastError());
}
```
